```python
import numpy as np

kernel_name = "gpu_gemm_a2a_m2048_k2048_n2048_f32_relu"


ABSMAX_RTOL = 0.001

COMPUTE_DTYPE = "float32"
EPILOGUE = "relu"
IN_QUANT = None
OUT_QUANT = None
IN_DTYPES = {"x": "float32", "w_mat": "float32"}
WORLD_SIZE = 8

def setup_inputs():

    x = np.random.randn(2048, 2048).astype(np.float32)
    w_mat = (np.random.randn(2048, 2048) / np.sqrt(2048)).astype(np.float32)
    return {"x": x, "w_mat": w_mat}

def _epilogue(y):
    return np.maximum(y, 0.0).astype(np.float32)

def reference(x, w_mat):
    y = (x @ w_mat).astype(np.float32)
    return _epilogue(y)
```

```cpp
#include <hip/hip_runtime.h>
#include <rccl/rccl.h>
#include <cstdio>
#include <cstdint>

#define HIP_CHECK(expr) do { hipError_t _e = (expr); if (_e != hipSuccess) { \
    fprintf(stderr, "HIP error %s at %s:%d\n", hipGetErrorString(_e), __FILE__, __LINE__); \
    return; } } while (0)

typedef _Float16 h8 __attribute__((ext_vector_type(8)));
typedef float f2 __attribute__((ext_vector_type(2)));
typedef float f4 __attribute__((ext_vector_type(4)));
typedef unsigned u4 __attribute__((ext_vector_type(4)));

constexpr int NR = 8;
constexpr int MR = 256;
constexpr int KD = 2048;
constexpr int ND = 2048;
constexpr int NB = 256;
constexpr int NWG = 256;
constexpr int NTHR = 512;
constexpr unsigned long long DEADLINE_TICKS = 50000000ull;

constexpr size_t WS_SEQ = 0;
constexpr size_t WS_FLAG = WS_SEQ + NWG * 128;
constexpr size_t WS_TMO = WS_FLAG + NWG * 128;
constexpr size_t WS_END = WS_TMO + 128;

struct Params {
    const float* x;
    const float* w;
    float* out[NR];
    char* ws[NR];
    int rank;
};

__device__ __forceinline__ h8 cvt8(f4 a, f4 b) {
    h8 r;
    r[0] = (_Float16)a[0]; r[1] = (_Float16)a[1]; r[2] = (_Float16)a[2]; r[3] = (_Float16)a[3];
    r[4] = (_Float16)b[0]; r[5] = (_Float16)b[1]; r[6] = (_Float16)b[2]; r[7] = (_Float16)b[3];
    return r;
}

__global__ void __launch_bounds__(NTHR, 2) fused_gemm_a2a(Params p) {
    __shared__ __attribute__((aligned(16))) float red[4][8][16][32];
    const int tid = threadIdx.x;
    const int lane = tid & 63;
    const int wave = __builtin_amdgcn_readfirstlane(tid >> 6);
    const int b = blockIdx.x;
    const int xg = b & 7;
    const int idx = b >> 3;
    const int mt = idx & 3;
    const int dst = ((idx >> 2) + p.rank) & 7;
    const int col0 = dst * NB + xg * 32;
    const int row0 = mt * 64;
    const int c = lane & 15, g = lane >> 4;

    unsigned* seqp = (unsigned*)(p.ws[p.rank] + WS_SEQ + (size_t)b * 128);
    unsigned call = 0;
    if (tid == 0) call = __hip_atomic_load(seqp, __ATOMIC_RELAXED, __HIP_MEMORY_SCOPE_AGENT) + 1;

    const int kb = wave * 256;
    const float* wp = p.w + (size_t)(kb + 4 * g) * ND + col0 + 2 * c;
    const float* xp = p.x + (size_t)(row0 + c) * KD + kb + 4 * g;

    h8 bf[8][2];
#pragma unroll
    for (int s = 0; s < 8; ++s) {
        f2 t[8];
#pragma unroll
        for (int j = 0; j < 8; ++j) {
            const int k = s * 32 + (j & 3) + (j >> 2) * 16;
            t[j] = *(const f2*)(wp + (size_t)k * ND);
        }
#pragma unroll
        for (int j = 0; j < 8; ++j) { bf[s][0][j] = (_Float16)t[j][0]; bf[s][1][j] = (_Float16)t[j][1]; }
    }
    float* outp = p.out[dst];
#pragma unroll
    for (int ch = 0; ch < 4; ++ch) {
        f4 acc0 = {0.f, 0.f, 0.f, 0.f}, acc1 = {0.f, 0.f, 0.f, 0.f};
#pragma unroll
        for (int s = 0; s < 8; ++s) {
            const float* q = xp + (size_t)ch * 16 * KD + s * 32;
            f4 a0 = *(const f4*)q, a1 = *(const f4*)(q + 16);
            h8 af = cvt8(a0, a1);
            acc0 = __builtin_amdgcn_mfma_f32_16x16x32_f16(af, bf[s][0], acc0, 0, 0, 0);
            acc1 = __builtin_amdgcn_mfma_f32_16x16x32_f16(af, bf[s][1], acc1, 0, 0, 0);
        }
#pragma unroll
        for (int r = 0; r < 4; ++r) {
            f2 v = {acc0[r], acc1[r]};
            *(f2*)&red[ch][wave][g * 4 + r][2 * c] = v;
        }
        __syncthreads();
        if ((wave >> 1) == ch) {
            const int t = tid & 127;
            const int rr = t >> 3, cc = (t & 7) * 4;
            f4 sum = *(const f4*)&red[ch][0][rr][cc];
#pragma unroll
            for (int wv = 1; wv < 8; ++wv) sum += *(const f4*)&red[ch][wv][rr][cc];
#pragma unroll
            for (int e = 0; e < 4; ++e) sum[e] = sum[e] < 0.f ? 0.f : sum[e];
            float* o = outp + (size_t)(p.rank * MR + row0 + ch * 16 + rr) * NB + xg * 32 + cc;
            *(f4*)o = sum;
        }
    }
    asm volatile("s_waitcnt vmcnt(0)" ::: "memory");
    __syncthreads();
    if (tid == 0) {
        __hip_atomic_store(seqp, call, __ATOMIC_RELAXED, __HIP_MEMORY_SCOPE_AGENT);
        if (dst != p.rank) {
            __builtin_amdgcn_fence(__ATOMIC_RELEASE, "");
            asm volatile("s_waitcnt vmcnt(0)" ::: "memory");
            unsigned* f = (unsigned*)(p.ws[dst] + WS_FLAG + (size_t)(p.rank * 32 + mt * 8 + xg) * 128);
            __hip_atomic_store(f, call, __ATOMIC_RELAXED, __HIP_MEMORY_SCOPE_SYSTEM);
        }
        if ((b >> 5) != p.rank) {
            unsigned* f = (unsigned*)(p.ws[p.rank] + WS_FLAG + (size_t)b * 128);
            unsigned* tmo = (unsigned*)(p.ws[p.rank] + WS_TMO);
            unsigned long long t0 = __builtin_amdgcn_s_memrealtime();
            bool bad = false;
            while (__hip_atomic_load(f, __ATOMIC_RELAXED, __HIP_MEMORY_SCOPE_SYSTEM) < call) {
                if (__hip_atomic_load(tmo, __ATOMIC_RELAXED, __HIP_MEMORY_SCOPE_AGENT) == call
                    || __builtin_amdgcn_s_memrealtime() - t0 > DEADLINE_TICKS) {
                    __hip_atomic_store(tmo, call, __ATOMIC_RELAXED, __HIP_MEMORY_SCOPE_AGENT);
                    bad = true;
                    break;
                }
                __builtin_amdgcn_s_sleep(1);
            }
            __builtin_amdgcn_fence(__ATOMIC_ACQUIRE, "");
            if (bad) {
                const int sr = b >> 5, t = b & 31;
                p.out[p.rank][(size_t)(sr * MR + (t >> 3) * 64) * NB + (t & 7) * 32] = __builtin_nanf("");
            }
        }
    }
}

extern "C" void kernel_launch(int rank, int nranks, ncclComm_t comm,
                              hipStream_t stream,
                              void* const* d_in, const int* in_sizes, int n_in,
                              void* d_out, int out_size,
                              const int* in_dtypes, int out_dtype,
                              void* const* d_out_all,
                              void* const* d_ws_all, size_t ws_bytes) {
    if (nranks != NR || (unsigned)rank >= (unsigned)NR || ws_bytes < WS_END || n_in < 2 ||
        in_sizes[0] != MR * KD || in_sizes[1] != KD * ND || out_size != NR * MR * NB) {
        fprintf(stderr, "kernel_launch: unexpected shapes\n");
        return;
    }
    Params p;
    p.x = (const float*)d_in[0];
    p.w = (const float*)d_in[1];
    for (int i = 0; i < NR; ++i) { p.out[i] = (float*)d_out_all[i]; p.ws[i] = (char*)d_ws_all[i]; }
    p.rank = rank;
    fused_gemm_a2a<<<NWG, NTHR, 0, stream>>>(p);
    HIP_CHECK(hipGetLastError());
}
```

```cpp
#include <hip/hip_runtime.h>
#include <rccl/rccl.h>
#include <cstdio>
#include <cstdint>

#define HIP_CHECK(expr) do { hipError_t _e = (expr); if (_e != hipSuccess) { \
    fprintf(stderr, "HIP error %s at %s:%d\n", hipGetErrorString(_e), __FILE__, __LINE__); \
    return; } } while (0)

typedef _Float16 h4 __attribute__((ext_vector_type(4)));
typedef _Float16 h8 __attribute__((ext_vector_type(8)));
typedef short s4 __attribute__((ext_vector_type(4)));
typedef float f4 __attribute__((ext_vector_type(4)));
typedef unsigned u4 __attribute__((ext_vector_type(4)));
#define LDS __attribute__((address_space(3)))

constexpr int NR = 8;
constexpr int MR = 256;
constexpr int KD = 2048;
constexpr int ND = 2048;
constexpr int NB = 256;
constexpr int NWG = 256;
constexpr int NTHR = 576;
constexpr unsigned long long DEADLINE_TICKS = 150000000ull;

constexpr size_t WS_SEQ = 0;
constexpr size_t WS_FLAG = WS_SEQ + NWG * 128;
constexpr size_t WS_TMO = WS_FLAG + NWG * 128;
constexpr size_t WS_PROF = 1 << 20;
constexpr size_t WS_XS = 2 << 20;
constexpr int XS_BYTES = MR * KD * 2;
constexpr size_t WS_END = WS_XS + 2 * XS_BYTES;

struct Params {
    const float* x;
    const float* w;
    char* wsl;
    int rank, pad;
    float* out[NR];
    char* ws[NR];
};

#define DEPTH0 4
#define DEPTH 6
#define BARN 6
#define BARMIN 13
#define FORCE_SLOW 0
#define TL 170
#define TL0 360
#define TL0S 260
#define FME 0xff
#define FML 0x01
#define GD 16
#define GL 32
#define GS 48
#define POLLN 16

#define STAMP(i) do {} while (0)
#define SB() __builtin_amdgcn_sched_barrier(0)

__device__ __forceinline__ h4 cvt4(u4 a) {
    f4 f = __builtin_bit_cast(f4, a);
    h4 r;
    r[0] = (_Float16)f[0]; r[1] = (_Float16)f[1]; r[2] = (_Float16)f[2]; r[3] = (_Float16)f[3];
    return r;
}
__device__ __forceinline__ unsigned umin3(unsigned a, unsigned b, unsigned c) { return min(min(a, b), c); }
__device__ __forceinline__ unsigned now() { return (unsigned)__builtin_amdgcn_s_memrealtime(); }

template <int I> struct IC { static constexpr int value = I; };
template <int I, int N, class F> __device__ __forceinline__ void static_for(F&& f) {
    if constexpr (I < N) { f(IC<I>{}); static_for<I + 1, N>(f); }
}

constexpr int NLOAD = 64;
constexpr int NW = 32;
constexpr int NCH = 4;
#define PK(a, b, c, d, e, f, g, h) ((a) | (b) << 4 | (c) << 8 | (d) << 12 | (e) << 16 | (f) << 20 | (g) << 24 | (h) << 28)
constexpr unsigned KO_EARLY = PK(0, 2, 4, 6, 1, 3, 5, 7), KO_LATE = PK(0, 1, 2, 4, 6, 3, 5, 7);
constexpr int XO[8] = {0, 4, 1, 5, 2, 6, 3, 7};
struct Sched {
    int a[NLOAD], b[NLOAD];
    int hi[NLOAD + 1];
    bool bar[NLOAD];
    int nb, bk[NLOAD];
};
constexpr Sched make_sched() {
    Sched s{};
    for (int i = 0; i < NW; ++i) { s.a[i] = i >> 2; s.b[i] = i & 3; }
    for (int i = 0; i < NLOAD - NW; ++i) { s.a[NW + i] = i >> 3; s.b[NW + i] = XO[i & 7]; }
    s.hi[0] = DEPTH0;
    for (int i = 1; i <= NLOAD; ++i) s.hi[i] = i + DEPTH > NLOAD ? NLOAD : i + DEPTH;
    for (int i = 0; i < NLOAD; ++i) {
        if (i >= NW && (i & 7) == 7) s.bk[s.nb++] = s.a[i];
        if (i >= BARMIN && (i - BARMIN) % BARN == 0 && i + BARN / 2 <= NW + 6) { s.bar[i] = true; s.bk[s.nb++] = -1; }
    }
    return s;
}
constexpr Sched SC = make_sched();

constexpr int LDS_R = 0;
constexpr int LDS_S = 65536;
constexpr int LDS_TOTAL = LDS_S + 8 * 1024;

__global__ void __launch_bounds__(NTHR, 3) fused_gemm_a2a(Params p) {
    __shared__ __attribute__((aligned(16))) char smem[LDS_TOTAL];
    const unsigned t0 = now();
    asm volatile("" :: "s"(p.x), "s"(p.w), "s"(p.wsl), "s"(p.rank));
    LDS char* lds = (LDS char*)smem;
    const int tid = threadIdx.x;
    const int lane = tid & 63;
    const int wave = __builtin_amdgcn_readfirstlane(tid >> 6) - 1;
    const int b = blockIdx.x;
    const int idx = b >> 3;
    const int mt = idx & 3;
    const int nt = idx >> 2;
    const int dst = (nt + p.rank) & 7;
    const int xc = b & 7;
    const int cg = (xc + nt) & 7;
    const int slowc = (3 - ((unsigned)(size_t)p.w >> 7)) & 7;
    const int ns = (slowc - xc) & 7;
    const int n1 = ns ? ns : 1;
    const int dn = nt >= n1 ? nt - n1 : nt - n1 + 7;
    const bool early = nt && !(dn & 1);
    const int ci = nt ? ((dn + 1) >> 1) & 3 : 0;
    const int rn = 2 * ci + !early;
    const int row0 = mt * 64;
    const int c = lane & 15, g = lane >> 4;
    const int r8 = lane >> 3, c8 = lane & 7;
    const __amdgpu_buffer_rsrc_t rs = __builtin_amdgcn_make_buffer_rsrc((void*)p.wsl, 0, (int)WS_END, 0x00020000);
    const int rot = (rn + 2 * mt) & 7;
    const unsigned kord = ((early ? KO_EARLY : KO_LATE) + rot * 0x11111111u) & 0x77777777u;

    if (wave >= 0) {
        const __amdgpu_buffer_rsrc_t rx = __builtin_amdgcn_make_buffer_rsrc((void*)p.x, 0, MR * KD * 4, 0x00020000);
        const __amdgpu_buffer_rsrc_t rw = __builtin_amdgcn_make_buffer_rsrc((void*)p.w, 0, KD * ND * 4, 0x00020000);
        const int lv = r8 * (ND * 4) + c8 * 16;
        const int fv = c * (KD * 4) + g * 16;
        const int col0 = dst * NB + cg * 32;
        const int kb = wave * 256;
        const int wsb = kb * (ND * 4) + col0 * 4;
        const int xsb = row0 * (KD * 4) + kb * 4;
        int hsb = (int)WS_XS + mt * 4 * 65536 + wave * 8192;
        LDS char* lb = lds + LDS_S + wave * 1024;
        const int bw = r8 * 64 + ((c8 ^ (r8 & 4)) << 3);
        const int br0 = (4 * g + (c >> 2)) * 64 + (c & 3) * 8 + (g & 1) * 32;
        const int br1 = br0 ^ 32;
        const int pw16 = (c >> 3) * 8192 + wave * 1024 + (c & 7) * 128 + ((g ^ (c & 7)) << 4);
        u4 buf[NLOAD];
        s4 bl[8][2], bh[8][2];
        f4 acc[NCH][2];
        unsigned vmin[NCH];
        unsigned help = 0;
        int nslow = 0;
        auto issue = [&](auto ic) __attribute__((always_inline)) {
            constexpr int i = decltype(ic)::value;
            constexpr int ia = SC.a[i], ib = SC.b[i];
            if constexpr (i < NW) {
                const int sp = (kord >> (4 * ia)) & 7;
                buf[i] = __builtin_amdgcn_raw_buffer_load_b128(rw, lv, wsb + (sp * 32 + ib * 8) * (ND * 4), 0);
            } else {
                buf[i] = __builtin_amdgcn_raw_buffer_load_b128(rs, lane * 16, hsb + ((kord >> (4 * ib)) & 7) * 1024 + ia * 65536, 16);
            }
        };
#define WF(j, h) __builtin_bit_cast(h8, __builtin_shufflevector(bl[j][h], bh[j][h], 0, 1, 2, 3, 4, 5, 6, 7))
        unsigned prev = __builtin_amdgcn_raw_buffer_load_b32(rs, lane & 0, (int)WS_SEQ + b * 128, 16);
        if (!early) {
            const int li = ci * 4 + mt;
            const int sk = xc * 8 + wave;
            const u4 lo = __builtin_amdgcn_raw_buffer_load_b128(rx, fv, (mt * 4 + ci) * 16 * (KD * 4) + sk * 128, 0);
            const u4 hi = __builtin_amdgcn_raw_buffer_load_b128(rx, fv + 64, (mt * 4 + ci) * 16 * (KD * 4) + sk * 128, 0);
            SB();
            __builtin_amdgcn_s_sleep(GD);
            constexpr int PS = (GS + 3) / 4;
            const int nn = (ns ? 3 : 4) * 4 * GL;
            const int per = (nn + 15) / 16;
            int q = li * per + lane, jm = q / GL, j = (jm >> 2) + (ns != 0), m = jm & 3, r = 64 - GL + q - jm * GL;
            bool on = q < nn;
            if (lane >= per) { q = li * PS + lane - per; j = 0; m = q / GS; r = 64 - GS + q - m * GS; on = ns && lane < per + PS && m < 4; }
            int en = n1 + ((2 * j) & 7);
            if (en > 7) en -= 7;
            const int row = ((2 * j + 2 * m) * 32 + r) & 255;
            const int go = on ? row * (ND * 4) + ((en + p.rank) & 7) * (NB * 4) + ((xc + en) & 7) * 128 : 0x40000000;
            help = __builtin_amdgcn_raw_buffer_load_b32(rw, go, kb * (ND * 4), 0);
            SB();
            u4 v = __builtin_bit_cast(u4, __builtin_shufflevector(cvt4(lo), cvt4(hi), 0, 1, 2, 3, 4, 5, 6, 7));
#pragma unroll
            for (int e = 0; e < 4; ++e) v[e] = v[e] ? v[e] : 0x80000000u;
            __builtin_amdgcn_raw_buffer_store_b128(v, rs, (int)WS_XS + (~prev & 1) * XS_BYTES + ((mt * 4 + ci) * 64 + sk) * 1024 + lane * 16, 0, 16);
            SB();
            STAMP(12);
            const int tl = ci ? TL : ns ? TL0 : TL0S;
            while ((int)(now() - t0) < tl) __builtin_amdgcn_s_sleep(1);
        }
        static_for<0, SC.hi[0]>(issue);
        SB();
        STAMP(1);
        static_for<0, NLOAD>([&](auto ic) __attribute__((always_inline)) {
            constexpr int i = decltype(ic)::value;
            constexpr int ia = SC.a[i], ib = SC.b[i];
            if constexpr (i == 1) { asm volatile("" : "+v"(prev)); hsb += (~__builtin_amdgcn_readfirstlane(prev) & 1) * XS_BYTES; }
            if constexpr (i < NW) {
                const h4 v = cvt4(buf[i]);
                if constexpr (i == 0) STAMP(2);
                if constexpr (i == 8) STAMP(3);
                if constexpr (i == NW - 1) STAMP(10);
                *(LDS h4*)(lb + (ib & 1) * 512 + bw) = v;
                if constexpr (SC.hi[i] <= NW && NW < SC.hi[i + 1]) STAMP(9);
                static_for<SC.hi[i], SC.hi[i + 1]>(issue);
                if constexpr (ib & 1) {
                    const s4 t0 = __builtin_amdgcn_ds_read_tr16_b64_v4i16((LDS s4*)(lb + br0));
                    const s4 t1 = __builtin_amdgcn_ds_read_tr16_b64_v4i16((LDS s4*)(lb + br1));
                    if constexpr (ib == 1) { bl[ia][0] = t0; bl[ia][1] = t1; }
                    else { bh[ia][0] = t0; bh[ia][1] = t1; }
                }
            } else {
                u4 u = buf[i];
                asm volatile("" : "+v"(u));
                static_for<SC.hi[i], SC.hi[i + 1]>(issue);
                if constexpr ((i & 7) == 0) { acc[ia][0] = f4{0.f, 0.f, 0.f, 0.f}; acc[ia][1] = acc[ia][0]; vmin[ia] = 1; }
                vmin[ia] = umin3(umin3(vmin[ia], u[0], u[1]), u[2], u[3]);
                SB();
                const h8 a = __builtin_bit_cast(h8, u);
                acc[ia][0] = __builtin_amdgcn_mfma_f32_16x16x32_f16(WF(ib, 0), a, acc[ia][0], 0, 0, 0);
                acc[ia][1] = __builtin_amdgcn_mfma_f32_16x16x32_f16(WF(ib, 1), a, acc[ia][1], 0, 0, 0);
                if constexpr ((i & 7) == 7) {
                    if (__builtin_expect(FORCE_SLOW || __builtin_amdgcn_ballot_w64(vmin[ia] == 0) != 0, 0)) {
                        acc[ia][0] = f4{0.f, 0.f, 0.f, 0.f}; acc[ia][1] = acc[ia][0];
#pragma unroll
                        for (int h = 0; h < 4; ++h) {
                            u4 lo[2], hi[2];
#pragma unroll
                            for (int q = 0; q < 2; ++q) {
                                const int sp = (kord >> (4 * (h * 2 + q))) & 7;
                                lo[q] = __builtin_amdgcn_raw_buffer_load_b128(rx, fv, xsb + sp * 128 + ia * 16 * (KD * 4), 0);
                                hi[q] = __builtin_amdgcn_raw_buffer_load_b128(rx, fv + 64, xsb + sp * 128 + ia * 16 * (KD * 4), 0);
                            }
#pragma unroll
                            for (int q = 0; q < 2; ++q) {
                                const int jj = h * 2 + q;
                                const h8 af = __builtin_shufflevector(cvt4(lo[q]), cvt4(hi[q]), 0, 1, 2, 3, 4, 5, 6, 7);
                                acc[ia][0] = __builtin_amdgcn_mfma_f32_16x16x32_f16(WF(jj, 0), af, acc[ia][0], 0, 0, 0);
                                acc[ia][1] = __builtin_amdgcn_mfma_f32_16x16x32_f16(WF(jj, 1), af, acc[ia][1], 0, 0, 0);
                            }
                            SB();
                        }
                        ++nslow;
                    }
                    *(LDS f4*)(lds + LDS_R + ia * 16384 + pw16) = acc[ia][0];
                    *(LDS f4*)(lds + LDS_R + ia * 16384 + (pw16 ^ 64)) = acc[ia][1];
                    asm volatile("s_waitcnt lgkmcnt(0)" ::: "memory");
                    __builtin_amdgcn_s_barrier();
                    STAMP(4 + ia);
                }
            }
            if constexpr (SC.bar[i]) __builtin_amdgcn_s_barrier();
            SB();
        });
        if (wave < 4) __builtin_amdgcn_raw_buffer_store_b128(u4{0, 0, 0, 0}, rs, (int)WS_XS + (prev & 1) * XS_BYTES + b * 4096 + wave * 1024 + lane * 16, 0, 16);
        asm volatile("" :: "v"(help));
    } else {
        __builtin_amdgcn_s_setprio(3);
        unsigned* seqp = (unsigned*)(p.wsl + WS_SEQ + (size_t)b * 128);
        const __amdgpu_buffer_rsrc_t ro = __builtin_amdgcn_make_buffer_rsrc((void*)p.out[dst], 0, NR * MR * NB * 4, 0x00020000);
        int ov[8];
#pragma unroll
        for (int pc = 0; pc < 8; ++pc) { ov[pc] = (p.rank * MR + row0 + pc * 8 + r8) * (NB * 4) + cg * 128 + c8 * 16; asm volatile("" : "+v"(ov[pc])); }
        const int rd = r8 * 128 + ((c8 ^ r8) << 4);
        unsigned* fdst = (unsigned*)(p.ws[dst] + WS_FLAG + (size_t)(p.rank * 32 + mt * 8 + cg) * 128);
        asm volatile("" : "+s"(fdst));
        __builtin_amdgcn_s_sleep(32);
        const unsigned prev = __hip_atomic_load(seqp, __ATOMIC_RELAXED, __HIP_MEMORY_SCOPE_AGENT);
        unsigned call = __builtin_amdgcn_readfirstlane(prev) + 1;
        asm volatile("" : "+s"(call));
        f4 res[8];
        const int fm = early ? FME : FML;
        int sent = 0;
        static_for<0, SC.nb>([&](auto kc) __attribute__((always_inline)) {
            constexpr int k = decltype(kc)::value;
            constexpr int ch = SC.bk[k];
            __builtin_amdgcn_s_barrier();
            asm volatile("" ::: "memory");
            if constexpr (ch >= 0) {
                f4 t[2][8];
                if constexpr (ch == 0) STAMP(8);
#pragma unroll
                for (int h = 0; h < 2; ++h)
#pragma unroll
                    for (int w_ = 0; w_ < 8; ++w_) t[h][w_] = *(LDS f4*)(lds + LDS_R + (2 * ch + h) * 8192 + rd + w_ * 1024);
                const bool fl = ch == NCH - 1 || ((fm >> ch) & 1);
                if (fl) {
#pragma unroll
                    for (int pc = 0; pc < 2 * ch; ++pc)
                        if (pc >= sent) __builtin_amdgcn_raw_buffer_store_b128(__builtin_bit_cast(u4, res[pc]), ro, ov[pc], 0, 17);
                    sent = 2 * ch + 2;
                }
                SB();
#pragma unroll
                for (int h = 0; h < 2; ++h) {
                    f4 sum = ((t[h][0] + t[h][1]) + (t[h][2] + t[h][3])) + ((t[h][4] + t[h][5]) + (t[h][6] + t[h][7]));
#pragma unroll
                    for (int e = 0; e < 4; ++e) sum[e] = __builtin_elementwise_maximum(sum[e], 0.f);
                    res[2 * ch + h] = sum;
                    if (fl) __builtin_amdgcn_raw_buffer_store_b128(__builtin_bit_cast(u4, sum), ro, ov[2 * ch + h], 0, 17);
                    SB();
                }
                STAMP(4 + ch);
            }
        });
        STAMP(11);
        if (lane == 0) {
            unsigned cv = call;
            asm volatile("s_waitcnt vmcnt(0)" : "+v"(cv) :: "memory");
            STAMP(12);
            __hip_atomic_store(fdst, cv, __ATOMIC_RELAXED, __HIP_MEMORY_SCOPE_SYSTEM);
            __hip_atomic_store(seqp, call, __ATOMIC_RELAXED, __HIP_MEMORY_SCOPE_AGENT);
            if ((b >> 5) != p.rank) {
                unsigned* f = (unsigned*)(p.wsl + WS_FLAG + (size_t)b * 128);
                unsigned* tmo = (unsigned*)(p.wsl + WS_TMO);
                unsigned long long tw = __builtin_amdgcn_s_memrealtime();
                bool bad = false;
                unsigned polls = 0;
                while (__hip_atomic_load(f, __ATOMIC_RELAXED, __HIP_MEMORY_SCOPE_SYSTEM) < call) {
                    if (++polls & (POLLN - 1)) continue;
                    if (__hip_atomic_load(tmo, __ATOMIC_RELAXED, __HIP_MEMORY_SCOPE_AGENT) == call
                        || __builtin_amdgcn_s_memrealtime() - tw > DEADLINE_TICKS) {
                        __hip_atomic_store(tmo, call, __ATOMIC_RELAXED, __HIP_MEMORY_SCOPE_AGENT);
                        bad = true;
                        break;
                    }
                    __builtin_amdgcn_s_sleep(1);
                }
                if (bad) {
                    const int sr = b >> 5, t = b & 31;
                    p.out[p.rank][(size_t)(sr * MR + (t >> 3) * 64) * NB + (t & 7) * 32] = __builtin_nanf("");
                    p.out[p.rank][(size_t)(p.rank * MR) * NB] = __builtin_nanf("");
                }
            }
        }
    }
}

extern "C" void kernel_launch(int rank, int nranks, ncclComm_t comm,
                              hipStream_t stream,
                              void* const* d_in, const int* in_sizes, int n_in,
                              void* d_out, int out_size,
                              const int* in_dtypes, int out_dtype,
                              void* const* d_out_all,
                              void* const* d_ws_all, size_t ws_bytes) {
    if (nranks != NR || (unsigned)rank >= (unsigned)NR || ws_bytes < WS_END || n_in < 2 ||
        in_sizes[0] != MR * KD || in_sizes[1] != KD * ND || out_size != NR * MR * NB) {
        fprintf(stderr, "kernel_launch: unexpected shapes\n");
        return;
    }
    Params p;
    p.x = (const float*)d_in[0];
    p.w = (const float*)d_in[1];
    for (int i = 0; i < NR; ++i) { p.out[i] = (float*)d_out_all[i]; p.ws[i] = (char*)d_ws_all[i]; }
    p.wsl = p.ws[rank];
    p.rank = rank; p.pad = 0;
    fused_gemm_a2a<<<NWG, NTHR, 0, stream>>>(p);
    HIP_CHECK(hipGetLastError());
}
```
